```python
import numpy as np

kernel_name = "gpu_gemm_ar_m2048_k2048_n2048_f32_relu"


ABSMAX_RTOL = 0.001

COMPUTE_DTYPE = "float32"
EPILOGUE = "relu"
IN_QUANT = None
OUT_QUANT = None
IN_DTYPES = {"x": "float32", "w_mat": "float32"}
WORLD_SIZE = 8

def setup_inputs():

    x = np.random.randn(2048, 2048).astype(np.float32)
    w_mat = (np.random.randn(2048, 2048) / np.sqrt(2048)).astype(np.float32)
    return {"x": x, "w_mat": w_mat}

def _epilogue(y):
    return np.maximum(y, 0.0).astype(np.float32)

def reference(x, w_mat):
    y = (x @ w_mat).astype(np.float32)
    return _epilogue(y)
```

```cpp
#include <hip/hip_runtime.h>
#include <rccl/rccl.h>
#include <cstdio>
#include <cstdint>

typedef unsigned u32x4 __attribute__((ext_vector_type(4)));
typedef float f32x4 __attribute__((ext_vector_type(4)));
typedef _Float16 f16x8 __attribute__((ext_vector_type(8)));

constexpr int NR = 8;
constexpr int MD = 2048, ND = 2048;
constexpr int KS = 256;
constexpr int NCH = 8;
constexpr int NBLK = 32;
constexpr int PIECE = 8192;
constexpr size_t STAGE_BYTES = (size_t)NR * NCH * NBLK * PIECE;
constexpr size_t OFF_FLAGS = 2 * STAGE_BYTES;
constexpr size_t OFF_CALL = OFF_FLAGS + NBLK * 128;
constexpr size_t OFF_TMO = OFF_CALL + 256 * 128;
constexpr size_t OFF_DBG = OFF_TMO + 128;
constexpr size_t WS_NEED = OFF_DBG + 256 * 128 * 8;
constexpr unsigned long long DEADLINE_TICKS = 30000000ull;
constexpr int NWAVES = 13, NTHREADS = NWAVES * 64;
constexpr int SLOT = 2 * PIECE;
constexpr int LDS_BYTES = NR * SLOT + 256;

struct Params {
  const float* x; const float* w; float* out;
  char* ws[NR];
  int rank;
};

#define LD16(dst, voff, sptr) asm volatile("global_load_dwordx4 %0, %1, %2" : "=v"(dst) : "v"(voff), "s"(sptr) : "memory")
#define ST16(voff, sptr, val, OFF) asm volatile("global_store_dwordx4 %0, %1, %2 offset:" #OFF " sc0 sc1\n\ts_nop 1" :: "v"(voff), "v"(val), "s"(sptr) : "memory")
#define WAITV8(N, r) asm volatile("s_waitcnt vmcnt(" #N ")" : "+v"(r[0]), "+v"(r[1]), "+v"(r[2]), "+v"(r[3]), "+v"(r[4]), "+v"(r[5]), "+v"(r[6]), "+v"(r[7]) :: "memory")
#define WAITV(N) asm volatile("s_waitcnt vmcnt(" #N ")" ::: "memory")

#define STAMP(i) do {} while (0)

__device__ __forceinline__ u32x4 pack8(float a0, float a1, float a2, float a3, float a4, float a5, float a6, float a7) {
  f16x8 h = {(_Float16)a0, (_Float16)a1, (_Float16)a2, (_Float16)a3, (_Float16)a4, (_Float16)a5, (_Float16)a6, (_Float16)a7};
  return __builtin_bit_cast(u32x4, h);
}

template <bool ISX> __device__ __forceinline__ void half_load(f32x4 (&r)[8], const float* base, unsigned voff, int t) {
  const int c = t >> 1, h = t & 1;
#pragma unroll
  for (int e = 0; e < 8; ++e) {
    const float* p = ISX ? base + (h * 64 + (e >> 1) * 16) * KS + c * 32 + (e & 1) * 4 : base + (c * 32 + e) * ND + h * 64;
    LD16(r[e], voff, p);
  }
}
template <bool ISX, int F> __device__ __forceinline__ u32x4 pack_frag(const f32x4 (&r)[8]) {
  return ISX ? pack8(r[2*F][0], r[2*F][1], r[2*F][2], r[2*F][3], r[2*F+1][0], r[2*F+1][1], r[2*F+1][2], r[2*F+1][3])
             : pack8(r[0][F], r[1][F], r[2][F], r[3][F], r[4][F], r[5][F], r[6][F], r[7][F]);
}
__device__ __forceinline__ void flag_store(unsigned* flag, unsigned v) {
  unsigned zero = 0;
  asm volatile("s_mov_b64 exec, 1\n\tglobal_store_dword %0, %1, %2 sc0 sc1\n\ts_mov_b64 exec, -1\n\ts_nop 1" :: "v"(zero), "v"(v), "s"(flag) : "memory");
}

#define PD 2
constexpr int NFR = NCH * 8;
constexpr bool flag_before(int j) { return j >= 7 + PD && (j - 7 - PD) % 8 == 0; }
constexpr int ops_after(int i, int j) {
  int n = 0;
  for (int k = i; k < j; ++k) {
    if (k > i) n += 1 + (flag_before(k) ? 1 : 0);
    if (k % 4 == 3 && k / 4 + 2 < 2 * NCH) n += 8;
  }
  return n;
}
#define WAITI8(N, r) asm volatile("s_waitcnt vmcnt(%8)" : "+v"(r[0]), "+v"(r[1]), "+v"(r[2]), "+v"(r[3]), "+v"(r[4]), "+v"(r[5]), "+v"(r[6]), "+v"(r[7]) : "i"(N) : "memory")
template <bool ISX, int J> __device__ __forceinline__ void push_step(f32x4 (&ra)[8], f32x4 (&rb)[8], const float* src, unsigned lo,
                                                                     char* dst, unsigned so, unsigned* flag, unsigned seq0,
                                                                     unsigned long long* stamps) {
  if constexpr (J < NFR) {
    constexpr int T = J / 4, F = J % 4;
    f32x4 (&r)[8] = (T & 1) ? rb : ra;
    if constexpr (J == 0) WAITI8(8, r);
    else if constexpr (J >= PD) { constexpr int N = ops_after(J - PD, J); WAITI8(N, r); }
    if constexpr (flag_before(J)) { flag_store(flag, seq0 + (J - 7 - PD) / 8 + 1); STAMP(1 + (J - 7 - PD) / 8); }
    char* p = dst + (size_t)(T >> 1) * (NBLK * PIECE) + (T & 1) * 4096;
    u32x4 v = pack_frag<ISX, F>(r);
    asm volatile("global_store_dwordx4 %0, %1, %2 offset:%3 sc0 sc1\n\ts_nop 1" :: "v"(so), "v"(v), "s"(p), "i"(F * 1024) : "memory");
    if constexpr (F == 3 && T + 2 < 2 * NCH) half_load<ISX>(r, src, lo, T + 2);
    push_step<ISX, J + 1>(ra, rb, src, lo, dst, so, flag, seq0, stamps);
  }
}

template <bool ISX> __device__ __forceinline__ void pusher(const Params& P, int g, int b, int lane, const unsigned* callp,
                                                           unsigned long long* stamps) {
  const float* src = ISX ? P.x + (size_t)(128 * b) * KS : P.w + 128 * (b - 16);
  const unsigned lo = ISX ? (unsigned)((lane & 15) * KS + 8 * (lane >> 4)) * 4u : (unsigned)(8 * (lane >> 4) * ND + 4 * (lane & 15)) * 4u;
  const unsigned so = lane * 16;
  f32x4 ra[8], rb[8];
  half_load<ISX>(ra, src, lo, 0); half_load<ISX>(rb, src, lo, 1);
  unsigned call;
  asm volatile("s_load_dword %0, %1, 0x0\n\ts_waitcnt lgkmcnt(0)\n\ts_barrier" : "=s"(call) : "s"(callp) : "memory");
  call += 1;
  const unsigned dest = (unsigned)(P.rank + (g >> 5)) & (NR - 1);
  char* dws = P.ws[dest];
  char* dst = dws + (size_t)(call & 1) * STAGE_BYTES + ((size_t)P.rank * NCH * NBLK + b) * PIECE;
  unsigned* flag = (unsigned*)(dws + OFF_FLAGS + b * 128) + P.rank;
  const unsigned seq0 = (call - 1) * NCH;
  push_step<ISX, 0>(ra, rb, src, lo, dst, so, flag, seq0, stamps);
  WAITV(0);
  flag_store(flag, seq0 + NCH); STAMP(1 + NCH - 1);
}

__global__ __launch_bounds__(NTHREADS) void gemm_ar_kernel(Params P) {
  extern __shared__ __attribute__((aligned(128))) char lds[];
  unsigned* full = (unsigned*)(lds + NR * SLOT);
  unsigned* freec = full + 8;
  unsigned* abortw = full + 16;
  const int g = blockIdx.x;
  const int wave = __builtin_amdgcn_readfirstlane(threadIdx.x >> 6);
  const int lane = threadIdx.x & 63;
  char* myws = P.ws[P.rank];
  unsigned* callp = (unsigned*)(myws + OFF_CALL + (size_t)g * 128);
  unsigned long long* stamps = nullptr;
  const unsigned long long t0 = __builtin_amdgcn_s_memrealtime();

  if (wave == 0) {
    STAMP(0);
    const int b = (g & 7) * 4 + ((g >> 3) & 3);
    if (b < 16) pusher<true>(P, g, b, lane, callp, stamps);
    else        pusher<false>(P, g, b, lane, callp, stamps);
    return;
  }

  const int xcd = g & 7, idx = g >> 3;
  const int ti = 4 * (xcd >> 1) + (idx >> 3), tj = 8 * (xcd & 1) + (idx & 7);
  unsigned call;
  asm volatile("s_load_dword %0, %1, 0x0\n\ts_waitcnt lgkmcnt(0)" : "=s"(call) : "s"(callp) : "memory");
  call += 1;
  if (wave == 1 && lane < 17) full[lane] = 0;
  asm volatile("s_waitcnt lgkmcnt(0)" ::: "memory");
  __builtin_amdgcn_s_barrier();
  asm volatile("" ::: "memory");

  if (wave >= 5) {
    const int s = (wave - 5 + P.rank) & (NR - 1);
    if (wave == 5 && lane == 0) *callp = call;
    const unsigned* flag = (const unsigned*)(myws + OFF_FLAGS + (size_t)((lane & 1) ? 16 + tj : ti) * 128) + s;
    const char* sx = myws + (size_t)(call & 1) * STAGE_BYTES + ((size_t)s * NCH * NBLK + ti) * PIECE + lane * 16;
    const char* sw = myws + (size_t)(call & 1) * STAGE_BYTES + ((size_t)s * NCH * NBLK + 16 + tj) * PIECE + lane * 16;
    char* slot = lds + s * SLOT;
    const unsigned seq0 = (call - 1) * NCH;
    unsigned known = seq0;
    bool dead = false;
    for (int c = 0; c < NCH; ++c) {
      while (!dead && (int)(known - (seq0 + c + 1)) < 0) {
        unsigned v = __hip_atomic_load(flag, __ATOMIC_RELAXED, __HIP_MEMORY_SCOPE_SYSTEM);
        unsigned v0 = __builtin_amdgcn_readlane(v, 0), v1 = __builtin_amdgcn_readlane(v, 1);
        known = (int)(v0 - v1) < 0 ? v0 : v1;
        if ((int)(known - (seq0 + c + 1)) >= 0) break;
        if (__builtin_amdgcn_s_memrealtime() - t0 > DEADLINE_TICKS) {
          dead = true;
          if (lane == 0) {
            __hip_atomic_store((unsigned*)(myws + OFF_TMO), call, __ATOMIC_RELAXED, __HIP_MEMORY_SCOPE_AGENT);
            __hip_atomic_store(abortw, 1u, __ATOMIC_RELAXED, __HIP_MEMORY_SCOPE_WORKGROUP);
          }
          break;
        }
        __builtin_amdgcn_s_sleep(4);
      }
      STAMP(16 + (wave - 5) * 8 + c);
      while (__hip_atomic_load(freec + s, __ATOMIC_ACQUIRE, __HIP_MEMORY_SCOPE_WORKGROUP) < 4u * c) __builtin_amdgcn_s_sleep(1);
      if (!dead) {
        const size_t co = (size_t)c * (NBLK * PIECE);
#pragma unroll
        for (int f = 0; f < 8; ++f)
          __builtin_amdgcn_global_load_lds((const __attribute__((address_space(1))) void*)(sx + co + f * 1024),
                                           (__attribute__((address_space(3))) void*)(slot + f * 1024), 16, 0, 17);
#pragma unroll
        for (int f = 0; f < 8; ++f)
          __builtin_amdgcn_global_load_lds((const __attribute__((address_space(1))) void*)(sw + co + f * 1024),
                                           (__attribute__((address_space(3))) void*)(slot + PIECE + f * 1024), 16, 0, 17);
      }
      WAITV(0);
      if (lane == 0) __hip_atomic_store(full + s, (unsigned)(c + 1), __ATOMIC_RELEASE, __HIP_MEMORY_SCOPE_WORKGROUP);
    }
    return;
  }

  const int wr = (wave - 1) >> 1, wc = (wave - 1) & 1;
  f32x4 acc[4][4];
#pragma unroll
  for (int m = 0; m < 4; ++m)
#pragma unroll
    for (int u = 0; u < 4; ++u) acc[m][u] = (f32x4){0.f, 0.f, 0.f, 0.f};
  unsigned consumed = 0;
  const char* abase = lds + wr * 4096 + lane * 16;
  const char* bbase = lds + PIECE + wc * 4096 + lane * 16;
  for (int it = 0; it < NR * NCH; ++it) {
    unsigned ready;
    for (;;) {
      unsigned v = __hip_atomic_load(full + (lane & 7), __ATOMIC_ACQUIRE, __HIP_MEMORY_SCOPE_WORKGROUP);
      ready = (unsigned)__builtin_amdgcn_ballot_w64(v > consumed) & 0xFFu;
      if (ready) break;
      __builtin_amdgcn_s_sleep(1);
    }
    const int s = __builtin_ctz(ready);
    f16x8 a[4], bq[4];
#pragma unroll
    for (int m = 0; m < 4; ++m) a[m] = *(const f16x8*)(abase + s * SLOT + m * 1024);
#pragma unroll
    for (int u = 0; u < 4; ++u) bq[u] = *(const f16x8*)(bbase + s * SLOT + u * 1024);
    asm volatile("s_waitcnt lgkmcnt(0)" : "+v"(a[0]), "+v"(a[1]), "+v"(a[2]), "+v"(a[3]), "+v"(bq[0]), "+v"(bq[1]), "+v"(bq[2]), "+v"(bq[3]) :: "memory");
    if (lane == 0) __hip_atomic_fetch_add(freec + s, 1u, __ATOMIC_RELEASE, __HIP_MEMORY_SCOPE_WORKGROUP);
    consumed += ((lane & 7) == s);
#pragma unroll
    for (int m = 0; m < 4; ++m)
#pragma unroll
      for (int u = 0; u < 4; ++u) acc[m][u] = __builtin_amdgcn_mfma_f32_16x16x32_f16(a[m], bq[u], acc[m][u], 0, 0, 0);
  }
  STAMP(80 + wave);
  const bool bad = __hip_atomic_load(abortw, __ATOMIC_RELAXED, __HIP_MEMORY_SCOPE_WORKGROUP) != 0;
  float* o = P.out + (size_t)(128 * ti + 64 * wr + 4 * (lane >> 4)) * ND + 128 * tj + 64 * wc + 4 * (lane & 15);
#pragma unroll
  for (int m = 0; m < 4; ++m)
#pragma unroll
    for (int r = 0; r < 4; ++r) {
      f32x4 v;
#pragma unroll
      for (int u = 0; u < 4; ++u) { float y = acc[m][u][r]; v[u] = bad ? __builtin_nanf("") : (y < 0.f ? 0.f : y); }
      *(f32x4*)(o + (size_t)(16 * m + r) * ND) = v;
    }
}

extern "C" void kernel_launch(int rank, int nranks, ncclComm_t comm, hipStream_t stream,
                              void* const* d_in, const int* in_sizes, int n_in,
                              void* d_out, int out_size, const int* in_dtypes, int out_dtype,
                              void* const* d_out_all, void* const* d_ws_all, size_t ws_bytes) {
  if (nranks != NR || (unsigned)rank >= (unsigned)NR || n_in != 2 || ws_bytes < WS_NEED ||
      in_sizes[0] != MD * KS || in_sizes[1] != KS * ND || out_size != MD * ND) {
    fprintf(stderr, "gemm_ar: unexpected configuration\n");
    return;
  }
  hipError_t e = hipFuncSetAttribute((const void*)gemm_ar_kernel, hipFuncAttributeMaxDynamicSharedMemorySize, LDS_BYTES);
  if (e != hipSuccess) { fprintf(stderr, "hipFuncSetAttribute: %s\n", hipGetErrorString(e)); return; }
  Params P;
  P.x = (const float*)d_in[0]; P.w = (const float*)d_in[1]; P.out = (float*)d_out; P.rank = rank;
  for (int p = 0; p < NR; ++p) P.ws[p] = (char*)d_ws_all[p];
  hipLaunchKernelGGL(gemm_ar_kernel, dim3(256), dim3(NTHREADS), LDS_BYTES, stream, P);
}
```

```cpp
#include <hip/hip_runtime.h>
#include <rccl/rccl.h>
#include <cstdio>
#include <cstdint>

typedef unsigned u32x4 __attribute__((ext_vector_type(4)));
typedef float f32x4 __attribute__((ext_vector_type(4)));
typedef _Float16 f16x8 __attribute__((ext_vector_type(8)));

constexpr int NR = 8;
constexpr int MD = 2048, ND = 2048;
constexpr int KS = 256;
constexpr int NCH = 8;
constexpr int NBLK = 32;
constexpr int PIECE = 8192;
constexpr size_t STAGE_BYTES = (size_t)NR * NCH * NBLK * PIECE;
constexpr size_t OFF_FLAGS = 2 * STAGE_BYTES;
constexpr size_t OFF_CALL = OFF_FLAGS + NBLK * 512;
constexpr size_t OFF_TICK = OFF_CALL + 256 * 128;
constexpr size_t OFF_TMO = OFF_TICK + NR * 256;
constexpr size_t OFF_DBG = OFF_TMO + 128;
constexpr size_t WS_NEED = OFF_DBG + 256 * 128 * 8;
constexpr unsigned long long DEADLINE_TICKS = 30000000ull;
constexpr int NWAVES = 13, NTHREADS = NWAVES * 64;
constexpr int SLOT = 2 * PIECE;
constexpr int LDS_BYTES = NR * SLOT + 256;

struct Params {
  const float* x; const float* w; float* out;
  char* ws[NR];
  int rank;
};

#define LD16(dst, voff, sptr) asm volatile("global_load_dwordx4 %0, %1, %2" : "=v"(dst) : "v"(voff), "s"(sptr) : "memory")
#define ST16(voff, sptr, val, OFF) asm volatile("global_store_dwordx4 %0, %1, %2 offset:" #OFF " sc0 sc1\n\ts_nop 1" :: "v"(voff), "v"(val), "s"(sptr) : "memory")
#define WAITV8(N, r) asm volatile("s_waitcnt vmcnt(" #N ")" : "+v"(r[0]), "+v"(r[1]), "+v"(r[2]), "+v"(r[3]), "+v"(r[4]), "+v"(r[5]), "+v"(r[6]), "+v"(r[7]) :: "memory")
#define WAITV(N) asm volatile("s_waitcnt vmcnt(" #N ")" ::: "memory")

#define STAMP(i) do {} while (0)

__device__ __forceinline__ u32x4 pack8(float a0, float a1, float a2, float a3, float a4, float a5, float a6, float a7) {
  f16x8 h = {(_Float16)a0, (_Float16)a1, (_Float16)a2, (_Float16)a3, (_Float16)a4, (_Float16)a5, (_Float16)a6, (_Float16)a7};
  return __builtin_bit_cast(u32x4, h);
}

template <bool ISX> __device__ __forceinline__ void half_load(f32x4 (&r)[8], const float* base, unsigned voff, int t) {
  const int c = t >> 1, h = t & 1;
#pragma unroll
  for (int e = 0; e < 8; ++e) {
    const float* p = ISX ? base + (h * 64 + (e >> 1) * 16) * KS + c * 32 + (e & 1) * 4 : base + (c * 32 + e) * ND + h * 64;
    LD16(r[e], voff, p);
  }
}
template <bool ISX, int F> __device__ __forceinline__ u32x4 pack_frag(const f32x4 (&r)[8]) {
  return ISX ? pack8(r[2*F][0], r[2*F][1], r[2*F][2], r[2*F][3], r[2*F+1][0], r[2*F+1][1], r[2*F+1][2], r[2*F+1][3])
             : pack8(r[0][F], r[1][F], r[2][F], r[3][F], r[4][F], r[5][F], r[6][F], r[7][F]);
}
__device__ __forceinline__ void flag_store(unsigned* flag, unsigned v) {
  unsigned zero = 0;
  asm volatile("s_mov_b64 exec, 1\n\tglobal_store_dword %0, %1, %2 sc0 sc1\n\ts_mov_b64 exec, -1\n\ts_nop 1" :: "v"(zero), "v"(v), "s"(flag) : "memory");
}

__device__ __forceinline__ unsigned draw(unsigned* tick) {
  unsigned zero = 0, one = 1, r;
  asm volatile("s_mov_b64 exec, 1\n\tglobal_atomic_add %0, %1, %2, %3 sc0 sc1\n\ts_mov_b64 exec, -1\n\ts_nop 1" : "=&v"(r) : "v"(zero), "v"(one), "s"(tick) : "memory");
  return r;
}

#define PD 2
#define NS 6
#define FASTN 24
constexpr int NDYN = (NCH - NS) * NBLK * 2;
constexpr int NFR = NS * 8 - 4;
constexpr bool flag_before(int j) { return j >= 7 + PD && (j - 7 - PD) % 8 == 0; }
constexpr int ops_after(int i, int j) {
  int n = 0;
  for (int k = i; k < j; ++k) {
    if (k > i) n += 1 + (flag_before(k) ? 1 : 0);
    if (k % 4 == 3 && k / 4 + 2 < 2 * NS) n += 8;
  }
  return n;
}
#define WAITI8(N, r) asm volatile("s_waitcnt vmcnt(%8)" : "+v"(r[0]), "+v"(r[1]), "+v"(r[2]), "+v"(r[3]), "+v"(r[4]), "+v"(r[5]), "+v"(r[6]), "+v"(r[7]) : "i"(N) : "memory")
template <bool ISX, int J> __device__ __forceinline__ void push_step(f32x4 (&ra)[8], f32x4 (&rb)[8], const float* src, unsigned lo,
                                                                     char* dst, unsigned so, unsigned* flag, unsigned seq0,
                                                                     unsigned* tick, unsigned& tk, unsigned long long* stamps) {
  if constexpr (J < NFR) {
    constexpr int T = J / 4, F = J % 4;
    f32x4 (&r)[8] = (T & 1) ? rb : ra;
    if constexpr (J == 0) WAITI8(8, r);
    else if constexpr (J >= PD) { constexpr int N = ops_after(J - PD, J); WAITI8(N, r); }
    if constexpr (flag_before(J)) { flag_store(flag, seq0 + (J - 7 - PD) / 8 + 1); STAMP(1 + (J - 7 - PD) / 8); }
    char* p = dst + (size_t)(T >> 1) * (NBLK * PIECE) + (T & 1) * 4096;
    u32x4 v = pack_frag<ISX, F>(r);
    if constexpr (J == NFR - 1) tk = draw(tick);
    asm volatile("global_store_dwordx4 %0, %1, %2 offset:%3 sc0 sc1\n\ts_nop 1" :: "v"(so), "v"(v), "s"(p), "i"(F * 1024) : "memory");
    if constexpr (F == 3 && T + 2 < 2 * NS) half_load<ISX>(r, src, lo, T + 2);
    push_step<ISX, J + 1>(ra, rb, src, lo, dst, so, flag, seq0, tick, tk, stamps);
  }
}

#define STP(v, p, OFF) asm volatile("global_store_dwordx4 %0, %1, %2 offset:" #OFF " sc0 sc1\n\ts_nop 1" :: "v"(so), "v"(v), "s"(p) : "memory")
__device__ __forceinline__ void pusher(const Params& P, int g, int b, int lane, const unsigned* callp, unsigned long long* stamps) {
  bool isx = b < 16;
  const float* src = isx ? P.x + (size_t)(128 * b) * KS : P.w + 128 * (b - 16);
  const unsigned lox = (unsigned)((lane & 15) * KS + 8 * (lane >> 4)) * 4u, low = (unsigned)(8 * (lane >> 4) * ND + 4 * (lane & 15)) * 4u;
  const unsigned so = lane * 16;
  const unsigned dest = (unsigned)(P.rank + (g >> 5)) & (NR - 1);
  char* dws = P.ws[dest];
  unsigned* tick = (unsigned*)(P.ws[P.rank] + OFF_TICK + (g >> 5) * 256);
  unsigned* flag = (unsigned*)(dws + OFF_FLAGS + b * 512 + P.rank * 64);
  f32x4 ra[8], r[8];
  unsigned call, tk = 0;
  char* stage;
  if (isx) {
    half_load<true>(ra, src, lox, 0); half_load<true>(r, src, lox, 1);
    asm volatile("s_load_dword %0, %1, 0x0\n\ts_waitcnt lgkmcnt(0)\n\ts_barrier" : "=s"(call) : "s"(callp) : "memory");
    call += 1;
    stage = dws + (size_t)(call & 1) * STAGE_BYTES + (size_t)P.rank * (NCH * NBLK * PIECE);
    push_step<true, 0>(ra, r, src, lox, stage + b * PIECE, so, flag, (call - 1) * NCH, tick, tk, stamps);
  } else {
    half_load<false>(ra, src, low, 0); half_load<false>(r, src, low, 1);
    asm volatile("s_load_dword %0, %1, 0x0\n\ts_waitcnt lgkmcnt(0)\n\ts_barrier" : "=s"(call) : "s"(callp) : "memory");
    call += 1;
    stage = dws + (size_t)(call & 1) * STAGE_BYTES + (size_t)P.rank * (NCH * NBLK * PIECE);
    push_step<false, 0>(ra, r, src, low, stage + b * PIECE, so, flag, (call - 1) * NCH, tick, tk, stamps);
  }
  const unsigned tbase = (call - 1) * (unsigned)(NDYN + 32);
  char* dst = stage + ((NS - 1) * NBLK + b) * PIECE + 4096;
  unsigned* pflag = flag; unsigned pval = (call - 1) * NCH + NS - 1;
  unsigned* cflag = flag; unsigned cval = (call - 1) * NCH + NS;
  bool fast = false;
  for (;;) {
    WAITV8(2, r);
    u32x4 v0, v1, v2, v3;
    if (isx) { v0 = pack_frag<true, 0>(r); v1 = pack_frag<true, 1>(r); v2 = pack_frag<true, 2>(r); v3 = pack_frag<true, 3>(r); }
    else     { v0 = pack_frag<false, 0>(r); v1 = pack_frag<false, 1>(r); v2 = pack_frag<false, 2>(r); v3 = pack_frag<false, 3>(r); }
    STP(v0, dst, 0);
    asm volatile("s_waitcnt vmcnt(1)" : "+v"(tk) :: "memory");
    flag_store(pflag, pval);
    STP(v1, dst, 1024);
    const unsigned t = __builtin_amdgcn_readfirstlane(tk) - tbase;
    if (t >= (unsigned)NDYN) { STP(v2, dst, 2048); STP(v3, dst, 3072); break; }
    const unsigned c = NS + (t >> 6), nb = (t >> 1) & 31, h = t & 1;
    isx = nb < 16;
    const float* nsrc = isx ? P.x + (128 * nb + 64 * h) * KS + 32 * c : P.w + 32 * c * ND + 128 * (nb - 16) + 64 * h;
    const unsigned lo = isx ? lox : low;
#pragma unroll
    for (int e = 0; e < 8; ++e) LD16(r[e], lo, nsrc + (isx ? (e >> 1) * 16 * KS + (e & 1) * 4 : e * ND));
    if (!fast) WAITV(10);
    STP(v2, dst, 2048);
    if (!fast) WAITV(9);
    tk = draw(tick);
    STP(v3, dst, 3072);
    fast = t >= (unsigned)(NDYN - FASTN);
    pflag = cflag; pval = cval;
    dst = stage + (c * NBLK + nb) * PIECE + h * 4096;
    cflag = (unsigned*)(dws + OFF_FLAGS + nb * 512 + P.rank * 64) + 2 * c + h; cval = call;
  }
  WAITV(0);
  flag_store(cflag, cval); STAMP(1 + NCH - 1);
}

__global__ __launch_bounds__(NTHREADS) void gemm_ar_kernel(Params P) {
  extern __shared__ __attribute__((aligned(128))) char lds[];
  unsigned* full = (unsigned*)(lds + NR * SLOT);
  unsigned* freec = full + 8;
  unsigned* abortw = full + 16;
  const int g = blockIdx.x;
  const int wave = __builtin_amdgcn_readfirstlane(threadIdx.x >> 6);
  const int lane = threadIdx.x & 63;
  char* myws = P.ws[P.rank];
  unsigned* callp = (unsigned*)(myws + OFF_CALL + (size_t)g * 128);
  unsigned long long* stamps = nullptr;
  const unsigned long long t0 = __builtin_amdgcn_s_memrealtime();

  if (wave == 0) {
    STAMP(0);
    const int b = (g & 7) * 4 + ((g >> 3) & 3);
    pusher(P, g, b, lane, callp, stamps);
    return;
  }

  const int xcd = g & 7, idx = g >> 3;
  const int ti = 4 * (xcd >> 1) + (idx >> 3), tj = 8 * (xcd & 1) + (idx & 7);
  unsigned call;
  asm volatile("s_load_dword %0, %1, 0x0\n\ts_waitcnt lgkmcnt(0)" : "=s"(call) : "s"(callp) : "memory");
  call += 1;
  if (wave == 1 && lane < 17) full[lane] = 0;
  asm volatile("s_waitcnt lgkmcnt(0)" ::: "memory");
  __builtin_amdgcn_s_barrier();
  asm volatile("" ::: "memory");

  if (wave >= 5) {
    const int s = (wave - 5 + P.rank) & (NR - 1);
    if (wave == 5 && lane == 0) *callp = call;
    const unsigned* flag = (const unsigned*)(myws + OFF_FLAGS + (size_t)((lane & 16) ? 16 + tj : ti) * 512 + s * 64) + (lane & 15);
    const char* sx = myws + (size_t)(call & 1) * STAGE_BYTES + ((size_t)s * NCH * NBLK + ti) * PIECE + lane * 16;
    const char* sw = myws + (size_t)(call & 1) * STAGE_BYTES + ((size_t)s * NCH * NBLK + 16 + tj) * PIECE + lane * 16;
    char* slot = lds + s * SLOT;
    const unsigned seq0 = (call - 1) * NCH;
    unsigned todo = 0, done = 0;
    bool dead = false;
    for (int it = 0; it < NCH; ++it) {
      while (!dead && !todo) {
        unsigned v = __hip_atomic_load(flag, __ATOMIC_RELAXED, __HIP_MEMORY_SCOPE_SYSTEM);
        int n0 = (int)(__builtin_amdgcn_readlane(v, 0) - seq0), n1 = (int)(__builtin_amdgcn_readlane(v, 16) - seq0);
        n0 = n0 < n1 ? n0 : n1; n0 = n0 < 0 ? 0 : (n0 > NS ? NS : n0);
        unsigned m = (unsigned)__builtin_amdgcn_ballot_w64((int)(v - call) >= 0);
        m &= m >> 16; m &= m >> 1;
        unsigned rdy = (1u << n0) - 1;
#pragma unroll
        for (int q = NS; q < NCH; ++q) rdy |= ((m >> (2 * q)) & 1u) << q;
        todo = rdy & ~done;
        if (todo) break;
        if (__builtin_amdgcn_s_memrealtime() - t0 > DEADLINE_TICKS) {
          dead = true;
          if (lane == 0) {
            __hip_atomic_store((unsigned*)(myws + OFF_TMO), call, __ATOMIC_RELAXED, __HIP_MEMORY_SCOPE_AGENT);
            __hip_atomic_store(abortw, 1u, __ATOMIC_RELAXED, __HIP_MEMORY_SCOPE_WORKGROUP);
          }
          break;
        }
        __builtin_amdgcn_s_sleep(4);
      }
      STAMP(16 + (wave - 5) * 8 + it);
      while (__hip_atomic_load(freec + s, __ATOMIC_ACQUIRE, __HIP_MEMORY_SCOPE_WORKGROUP) < 4u * it) __builtin_amdgcn_s_sleep(1);
      if (!dead) {
        const int c = __builtin_ctz(todo) & (NCH - 1);
        done |= 1u << c; todo &= todo - 1;
        const size_t co = (size_t)c * (NBLK * PIECE);
#pragma unroll
        for (int f = 0; f < 8; ++f)
          __builtin_amdgcn_global_load_lds((const __attribute__((address_space(1))) void*)(sx + co + f * 1024),
                                           (__attribute__((address_space(3))) void*)(slot + f * 1024), 16, 0, 17);
#pragma unroll
        for (int f = 0; f < 8; ++f)
          __builtin_amdgcn_global_load_lds((const __attribute__((address_space(1))) void*)(sw + co + f * 1024),
                                           (__attribute__((address_space(3))) void*)(slot + PIECE + f * 1024), 16, 0, 17);
      }
      WAITV(0);
      if (lane == 0) __hip_atomic_store(full + s, (unsigned)(it + 1), __ATOMIC_RELEASE, __HIP_MEMORY_SCOPE_WORKGROUP);
    }
    return;
  }

  const int wr = (wave - 1) >> 1, wc = (wave - 1) & 1;
  f32x4 acc[4][4];
#pragma unroll
  for (int m = 0; m < 4; ++m)
#pragma unroll
    for (int u = 0; u < 4; ++u) acc[m][u] = (f32x4){0.f, 0.f, 0.f, 0.f};
  unsigned consumed = 0;
  const char* abase = lds + wr * 4096 + lane * 16;
  const char* bbase = lds + PIECE + wc * 4096 + lane * 16;
  for (int it = 0; it < NR * NCH; ++it) {
    unsigned ready;
    for (;;) {
      unsigned v = __hip_atomic_load(full + (lane & 7), __ATOMIC_ACQUIRE, __HIP_MEMORY_SCOPE_WORKGROUP);
      ready = (unsigned)__builtin_amdgcn_ballot_w64(v > consumed) & 0xFFu;
      if (ready) break;
      __builtin_amdgcn_s_sleep(1);
    }
    const int s = __builtin_ctz(ready);
    f16x8 a[4], bq[4];
#pragma unroll
    for (int m = 0; m < 4; ++m) a[m] = *(const f16x8*)(abase + s * SLOT + m * 1024);
#pragma unroll
    for (int u = 0; u < 4; ++u) bq[u] = *(const f16x8*)(bbase + s * SLOT + u * 1024);
    asm volatile("s_waitcnt lgkmcnt(0)" : "+v"(a[0]), "+v"(a[1]), "+v"(a[2]), "+v"(a[3]), "+v"(bq[0]), "+v"(bq[1]), "+v"(bq[2]), "+v"(bq[3]) :: "memory");
    if (lane == 0) __hip_atomic_fetch_add(freec + s, 1u, __ATOMIC_RELEASE, __HIP_MEMORY_SCOPE_WORKGROUP);
    consumed += ((lane & 7) == s);
#pragma unroll
    for (int m = 0; m < 4; ++m)
#pragma unroll
      for (int u = 0; u < 4; ++u) acc[m][u] = __builtin_amdgcn_mfma_f32_16x16x32_f16(a[m], bq[u], acc[m][u], 0, 0, 0);
  }
  STAMP(80 + wave);
  const bool bad = __hip_atomic_load(abortw, __ATOMIC_RELAXED, __HIP_MEMORY_SCOPE_WORKGROUP) != 0;
  float* o = P.out + (size_t)(128 * ti + 64 * wr + 4 * (lane >> 4)) * ND + 128 * tj + 64 * wc + 4 * (lane & 15);
#pragma unroll
  for (int m = 0; m < 4; ++m)
#pragma unroll
    for (int r = 0; r < 4; ++r) {
      f32x4 v;
#pragma unroll
      for (int u = 0; u < 4; ++u) { float y = acc[m][u][r]; v[u] = bad ? __builtin_nanf("") : (y < 0.f ? 0.f : y); }
      asm volatile("global_store_dwordx4 %0, %1, off sc0 sc1 nt\n\ts_nop 1" :: "v"(o + (size_t)(16 * m + r) * ND), "v"(v) : "memory");
    }
}

extern "C" void kernel_launch(int rank, int nranks, ncclComm_t comm, hipStream_t stream,
                              void* const* d_in, const int* in_sizes, int n_in,
                              void* d_out, int out_size, const int* in_dtypes, int out_dtype,
                              void* const* d_out_all, void* const* d_ws_all, size_t ws_bytes) {
  if (nranks != NR || (unsigned)rank >= (unsigned)NR || n_in != 2 || ws_bytes < WS_NEED ||
      in_sizes[0] != MD * KS || in_sizes[1] != KS * ND || out_size != MD * ND) {
    fprintf(stderr, "gemm_ar: unexpected configuration\n");
    return;
  }
  hipError_t e = hipFuncSetAttribute((const void*)gemm_ar_kernel, hipFuncAttributeMaxDynamicSharedMemorySize, LDS_BYTES);
  if (e != hipSuccess) { fprintf(stderr, "hipFuncSetAttribute: %s\n", hipGetErrorString(e)); return; }
  Params P;
  P.x = (const float*)d_in[0]; P.w = (const float*)d_in[1]; P.out = (float*)d_out; P.rank = rank;
  for (int p = 0; p < NR; ++p) P.ws[p] = (char*)d_ws_all[p];
  hipLaunchKernelGGL(gemm_ar_kernel, dim3(256), dim3(NTHREADS), LDS_BYTES, stream, P);
}
```

```cpp
#include <hip/hip_runtime.h>
#include <rccl/rccl.h>
#include <cstdio>
#include <cstdint>

typedef unsigned u32x4 __attribute__((ext_vector_type(4)));
typedef float f32x4 __attribute__((ext_vector_type(4)));
typedef _Float16 f16x8 __attribute__((ext_vector_type(8)));

constexpr int NR = 8;
constexpr int MD = 2048, ND = 2048;
constexpr int KS = 256;
constexpr int NCH = 8;
constexpr int NBLK = 32;
constexpr int PIECE = 8192;
constexpr size_t STAGE_BYTES = (size_t)NR * NCH * NBLK * PIECE;
constexpr size_t OFF_FLAGS = 2 * STAGE_BYTES;
constexpr size_t OFF_CALL = OFF_FLAGS + NBLK * 512;
constexpr size_t OFF_TICK = OFF_CALL + 256 * 128;
constexpr size_t OFF_TMO = OFF_TICK + NR * 256;
constexpr size_t OFF_DBG = OFF_TMO + 128;
constexpr size_t WS_NEED = OFF_DBG + 256 * 128 * 8;
constexpr unsigned long long DEADLINE_TICKS = 30000000ull;
constexpr int NWAVES = 13, NTHREADS = NWAVES * 64;
constexpr int SLOT = 2 * PIECE;
constexpr int LDS_BYTES = NR * SLOT + 256;

struct Params {
  const float* x; const float* w; float* out;
  char* ws[NR];
  int rank;
};

#define LD16(dst, voff, sptr) asm volatile("global_load_dwordx4 %0, %1, %2" : "=v"(dst) : "v"(voff), "s"(sptr) : "memory")
#define ST16(voff, sptr, val, OFF) asm volatile("global_store_dwordx4 %0, %1, %2 offset:" #OFF " sc0 sc1\n\ts_nop 1" :: "v"(voff), "v"(val), "s"(sptr) : "memory")
#define WAITV8(N, r) asm volatile("s_waitcnt vmcnt(" #N ")" : "+v"(r[0]), "+v"(r[1]), "+v"(r[2]), "+v"(r[3]), "+v"(r[4]), "+v"(r[5]), "+v"(r[6]), "+v"(r[7]) :: "memory")
#define WAITV(N) asm volatile("s_waitcnt vmcnt(" #N ")" ::: "memory")

#define STAMP(i) do {} while (0)

__device__ __forceinline__ u32x4 pack8(float a0, float a1, float a2, float a3, float a4, float a5, float a6, float a7) {
  f16x8 h = {(_Float16)a0, (_Float16)a1, (_Float16)a2, (_Float16)a3, (_Float16)a4, (_Float16)a5, (_Float16)a6, (_Float16)a7};
  return __builtin_bit_cast(u32x4, h);
}

__device__ __forceinline__ void half_load(f32x4 (&r)[8], const Params& P, unsigned nb, unsigned c, unsigned h, unsigned lox, unsigned low) {
  const bool isx = nb < 16;
  const float* src = isx ? P.x + (128 * nb + 64 * h) * KS + 32 * c : P.w + 32 * c * ND + 128 * (nb - 16) + 64 * h;
  const unsigned lo = isx ? lox : low;
#pragma unroll
  for (int e = 0; e < 8; ++e) LD16(r[e], lo, src + (isx ? (e >> 1) * 16 * KS + (e & 1) * 4 : e * ND));
}
template <bool ISX, int F> __device__ __forceinline__ u32x4 pack_frag(const f32x4 (&r)[8]) {
  return ISX ? pack8(r[2*F][0], r[2*F][1], r[2*F][2], r[2*F][3], r[2*F+1][0], r[2*F+1][1], r[2*F+1][2], r[2*F+1][3])
             : pack8(r[0][F], r[1][F], r[2][F], r[3][F], r[4][F], r[5][F], r[6][F], r[7][F]);
}
__device__ __forceinline__ void flag_store(unsigned* flag, unsigned v) {
  unsigned zero = 0;
  asm volatile("s_mov_b64 exec, 1\n\tglobal_store_dword %0, %1, %2 sc0 sc1\n\ts_mov_b64 exec, -1\n\ts_nop 1" :: "v"(zero), "v"(v), "s"(flag) : "memory");
}

__device__ __forceinline__ unsigned draw(unsigned* tick) {
  unsigned zero = 0, one = 1, r;
  asm volatile("s_mov_b64 exec, 1\n\tglobal_atomic_add %0, %1, %2, %3 sc0 sc1\n\ts_mov_b64 exec, -1\n\ts_nop 1" : "=&v"(r) : "v"(zero), "v"(one), "s"(tick) : "memory");
  return r;
}

#define PSLEEP 4
#define NS 6
#define FASTN 24
constexpr int NDYN = (NCH - NS) * NBLK * 2;
constexpr unsigned HDR_RAW = 32;

typedef unsigned short u16x2 __attribute__((ext_vector_type(2)));
typedef volatile __attribute__((address_space(3))) unsigned short lds_u16;
typedef volatile __attribute__((address_space(3))) u32x4 lds_u32x4;
__device__ __forceinline__ unsigned bfi(unsigned m, unsigned a, unsigned b) { return (a & m) | (b & ~m); }
struct Enc { unsigned h[16], base, q0, q1; };
struct Out { u32x4 v[4]; unsigned list, hdr; };
__device__ __forceinline__ void cvt16(unsigned (&h)[16], const f32x4 (&r)[8], bool isx) {
  u32x4 v0, v1, v2, v3;
  if (isx) { v0 = pack_frag<true, 0>(r); v1 = pack_frag<true, 1>(r); v2 = pack_frag<true, 2>(r); v3 = pack_frag<true, 3>(r); }
  else     { v0 = pack_frag<false, 0>(r); v1 = pack_frag<false, 1>(r); v2 = pack_frag<false, 2>(r); v3 = pack_frag<false, 3>(r); }
#pragma unroll
  for (int j = 0; j < 4; ++j) { h[j] = v0[j]; h[4 + j] = v1[j]; h[8 + j] = v2[j]; h[12 + j] = v3[j]; }
  asm volatile("" : "+v"(h[0]), "+v"(h[1]), "+v"(h[2]), "+v"(h[3]), "+v"(h[4]), "+v"(h[5]), "+v"(h[6]), "+v"(h[7]),
                    "+v"(h[8]), "+v"(h[9]), "+v"(h[10]), "+v"(h[11]), "+v"(h[12]), "+v"(h[13]), "+v"(h[14]), "+v"(h[15]));
}
__device__ __forceinline__ void enc_a(Enc& E) {
  u16x2 mx = {0, 0};
#pragma unroll
  for (int j = 0; j < 16; ++j) mx = __builtin_elementwise_max(mx, __builtin_bit_cast(u16x2, E.h[j] & 0x7FFF7FFFu));
  const unsigned e = (unsigned)(mx[0] > mx[1] ? mx[0] : mx[1]) >> 10;
  unsigned top = 0;
#pragma unroll
  for (int bit = 4; bit >= 0; --bit) {
    const unsigned cand = top | (1u << bit);
    if (__builtin_popcountll(__builtin_amdgcn_ballot_w64(e >= cand)) >= 16) top = cand;
  }
  E.base = top < 7 ? 0 : top - 7;
}
__device__ __forceinline__ void enc_b(Enc& E, Out& O) {
  const u16x2 bb = {(unsigned short)(E.base << 10), (unsigned short)(E.base << 10)};
  unsigned t[16], q0 = 0, q1 = 0;
#pragma unroll
  for (int j = 0; j < 16; ++j) {
    const unsigned d = __builtin_bit_cast(unsigned, (u16x2)(__builtin_bit_cast(u16x2, E.h[j]) - bb));
    t[j] = bfi(0x80008000u, E.h[j], d);
    if (j < 8) q0 = bfi(0xC000C000u, d << 1, q0 >> 2); else q1 = bfi(0xC000C000u, d << 1, q1 >> 2);
  }
  const unsigned g0 = bfi(0x40004000u, t[14] >> 1, t[14] << 1), g1 = bfi(0x40004000u, t[15] >> 1, t[15] << 1);
#pragma unroll
  for (int k = 0; k < 14; ++k) O.v[k >> 2][k & 3] = bfi(0x60006000u, (k < 7 ? g0 : g1) << (12 - 2 * (k % 7)), t[k]);
  O.v[3][2] = 0; O.v[3][3] = 0;
  E.q0 = q0; E.q1 = q1;
}
__device__ __forceinline__ void enc_c(Enc& E, Out& O, int lane, lds_u16* scr) {
  unsigned m = ((E.q0 | (E.q0 >> 1)) & 0x55555555u) | (((E.q1 | (E.q1 >> 1)) & 0x55555555u) << 1);
  scr[lane] = 0;
  unsigned total = 0;
  for (;;) {
    const unsigned long long bal = __builtin_amdgcn_ballot_w64(m != 0);
    if (bal == 0) break;
    const unsigned pos = total + __builtin_amdgcn_mbcnt_hi((unsigned)(bal >> 32), __builtin_amdgcn_mbcnt_lo((unsigned)bal, 0));
    total += __builtin_popcountll(bal);
    if (total > 64) break;
    if (m != 0) {
      const unsigned p = __builtin_ctz(m), j = ((p & 15) >> 1) + 8 * (p & 1);
      const unsigned bits = (((p & 1) ? E.q1 : E.q0) >> (p & 30)) & 3u;
      scr[pos] = (unsigned short)((j >> 2) * 512 + lane * 8 + (j & 3) * 2 + (p >> 4) + (bits << 13));
      m &= m - 1;
    }
  }
  total = __builtin_amdgcn_readfirstlane(total);
  O.list = scr[lane];
  O.hdr = E.base;
  if (total > 64) {
    O.hdr = HDR_RAW;
#pragma unroll
    for (int j = 0; j < 16; ++j) O.v[j >> 2][j & 3] = E.h[j];
  }
}
#define STP(v, p, OFF) asm volatile("global_store_dwordx4 %0, %1, %2 offset:" #OFF " sc0 sc1\n\ts_nop 1" :: "v"(so), "v"(v), "s"(p) : "memory")
#define PIN() __builtin_amdgcn_sched_barrier(0)
__device__ __forceinline__ void store_tail(const Out& O, char* dst, unsigned so) {
  if (O.hdr == HDR_RAW) { STP(O.v[3], dst, 3072); WAITV(1); }
  else {
    unsigned long long lo = (unsigned long long)O.v[3][0] | ((unsigned long long)O.v[3][1] << 32);
    asm volatile("global_store_dwordx2 %0, %1, %2 offset:3072 sc0 sc1\n\tglobal_store_short %3, %4, %2 offset:3584 sc0 sc1\n\ts_nop 1"
                 :: "v"(so >> 1), "v"(lo), "s"(dst), "v"(so >> 3), "v"(O.list) : "memory");
  }
}
template <int ODD> __device__ __forceinline__ void send_half(const Params& P, unsigned b, unsigned c, const Out& C, Out& N, f32x4 (&r)[8],
                                                             char* dst, unsigned* flag, unsigned fval, int lane, unsigned lox, unsigned low,
                                                             lds_u16* scr) {
  const unsigned so = lane * 16;
  Enc E;
  WAITV(2);
  STP(C.v[0], dst, 0); WAITV8(4, r); PIN();
  cvt16(E.h, r, b < 16);
  half_load(r, P, b, c + 1, ODD, lox, low);
  enc_a(E); PIN();
  WAITV(9);
  if (!ODD && c > 0) flag_store(flag + 2 * (c - 1), fval);
  STP(C.v[1], dst, 1024); PIN();
  enc_b(E, N); PIN();
  if (ODD) WAITV(9); else WAITV(10);
  STP(C.v[2], dst, 2048); PIN();
  enc_c(E, N, lane, scr); PIN();
  WAITV(1);
  store_tail(C, dst, so);
}

__device__ __forceinline__ void pusher(const Params& P, int g, unsigned b, int lane, const unsigned* callp, lds_u16* scr,
                                       unsigned long long* stamps) {
  bool isx = b < 16;
  const unsigned lox = (unsigned)((lane & 15) * KS + 8 * (lane >> 4)) * 4u, low = (unsigned)(8 * (lane >> 4) * ND + 4 * (lane & 15)) * 4u;
  const unsigned so = lane * 16;
  const unsigned dest = (unsigned)(P.rank + (g >> 5)) & (NR - 1);
  char* dws = P.ws[dest];
  unsigned* tick = (unsigned*)(P.ws[P.rank] + OFF_TICK + (g >> 5) * 256);
  unsigned* flag = (unsigned*)(dws + OFF_FLAGS + b * 512 + P.rank * 64);
  f32x4 ra[8], r[8];
  unsigned call, tk;
  half_load(ra, P, b, 0, 0, lox, low); half_load(r, P, b, 0, 1, lox, low);
  asm volatile("s_load_dword %0, %1, 0x0\n\ts_waitcnt lgkmcnt(0)\n\ts_barrier" : "=s"(call) : "s"(callp) : "memory");
  call += 1;
  char* stage = dws + (size_t)(call & 1) * STAGE_BYTES + (size_t)P.rank * (NCH * NBLK * PIECE);
  const unsigned tag = call << 12;
  Out A, B;
  {
    Enc E;
    WAITV8(8, ra);
    cvt16(E.h, ra, isx);
    A.hdr = HDR_RAW; A.list = 0;
#pragma unroll
    for (int j = 0; j < 16; ++j) A.v[j >> 2][j & 3] = E.h[j];
    WAITV8(0, r);
  }
  unsigned fval = 0;
  char* dst = stage + b * PIECE;
  for (unsigned c = 0; c < NS - 1; ++c) {
    send_half<0>(P, b, c, A, B, r, dst, flag, fval, lane, lox, low, scr);
    if (c == 1) STAMP(1);
    fval = tag | A.hdr | (B.hdr << 6);
    send_half<1>(P, b, c, B, A, r, dst + 4096, flag, 0, lane, lox, low, scr);
    dst += NBLK * PIECE;
  }
  WAITV(2); STP(A.v[0], dst, 0);
  WAITV(1); flag_store(flag + 2 * (NS - 2), fval); STP(A.v[1], dst, 1024);
  WAITV(2); STP(A.v[2], dst, 2048);
  WAITV(1); tk = draw(tick); store_tail(A, dst, so);
  const unsigned tbase = (call - 1) * (unsigned)(NDYN + 32);
  dst += 4096;
  unsigned* pflag = flag + 2 * (NS - 2); unsigned pval = fval;
  unsigned* cflag = flag + 2 * (NS - 1); unsigned cval = tag | A.hdr | (HDR_RAW << 6);
  bool fast = false;
  for (;;) {
    WAITV8(2, r);
    u32x4 v0, v1, v2, v3;
    if (isx) { v0 = pack_frag<true, 0>(r); v1 = pack_frag<true, 1>(r); v2 = pack_frag<true, 2>(r); v3 = pack_frag<true, 3>(r); }
    else     { v0 = pack_frag<false, 0>(r); v1 = pack_frag<false, 1>(r); v2 = pack_frag<false, 2>(r); v3 = pack_frag<false, 3>(r); }
    STP(v0, dst, 0);
    asm volatile("s_waitcnt vmcnt(1)" : "+v"(tk) :: "memory");
    flag_store(pflag, pval);
    STP(v1, dst, 1024);
    const unsigned t = __builtin_amdgcn_readfirstlane(tk) - tbase;
    if (t >= (unsigned)NDYN) { STP(v2, dst, 2048); STP(v3, dst, 3072); break; }
    const unsigned c = NS + (t >> 6), nb = (t >> 1) & 31, h = t & 1;
    isx = nb < 16;
    half_load(r, P, nb, c, h, lox, low);
    if (!fast) WAITV(10);
    STP(v2, dst, 2048);
    if (!fast) WAITV(9);
    tk = draw(tick);
    STP(v3, dst, 3072);
    fast = t >= (unsigned)(NDYN - FASTN);
    pflag = cflag; pval = cval;
    dst = stage + (c * NBLK + nb) * PIECE + h * 4096;
    cflag = (unsigned*)(dws + OFF_FLAGS + nb * 512 + P.rank * 64) + 2 * c + h; cval = tag | HDR_RAW | (HDR_RAW << 6);
  }
  WAITV(0);
  flag_store(cflag, cval); STAMP(1 + NCH - 1);
}

__device__ __forceinline__ unsigned ready_mask(unsigned v, unsigned tag) {
  unsigned m = (unsigned)__builtin_amdgcn_ballot_w64(((v ^ tag) >> 12) == 0);
  m &= m >> 16; m &= (m >> 1) | ((1u << (2 * NS)) - 1);
  unsigned rdy = 0;
#pragma unroll
  for (int q = 0; q < NCH; ++q) rdy |= ((m >> (2 * q)) & 1u) << q;
  return rdy;
}

__global__ __launch_bounds__(NTHREADS) void gemm_ar_kernel(Params P) {
  extern __shared__ __attribute__((aligned(128))) char lds[];
  unsigned* full = (unsigned*)(lds + NR * SLOT);
  unsigned* freec = full + 8;
  unsigned* abortw = full + 16;
  const int g = blockIdx.x;
  const int wave = __builtin_amdgcn_readfirstlane(threadIdx.x >> 6);
  const int lane = threadIdx.x & 63;
  char* myws = P.ws[P.rank];
  unsigned* callp = (unsigned*)(myws + OFF_CALL + (size_t)g * 128);
  unsigned long long* stamps = nullptr;
  const unsigned long long t0 = __builtin_amdgcn_s_memrealtime();

  if (wave == 0) {
    STAMP(0);
    const int b = (g & 7) * 4 + ((g >> 3) & 3);
    pusher(P, g, b, lane, callp, (lds_u16*)(lds + NR * SLOT + 128), stamps);
    return;
  }

  const int xcd = g & 7, idx = g >> 3;
  const int ti = 4 * (xcd >> 1) + (idx >> 3), tj = 8 * (xcd & 1) + (idx & 7);
  unsigned call;
  asm volatile("s_load_dword %0, %1, 0x0\n\ts_waitcnt lgkmcnt(0)" : "=s"(call) : "s"(callp) : "memory");
  call += 1;
  if (wave == 1 && lane < 17) full[lane] = 0;
  asm volatile("s_waitcnt lgkmcnt(0)" ::: "memory");
  __builtin_amdgcn_s_barrier();
  asm volatile("" ::: "memory");

  if (wave >= 5) {
    const int s = (wave - 4 + P.rank) & (NR - 1);
    if (wave == 5 && lane == 0) *callp = call;
    const unsigned* flag = (const unsigned*)(myws + OFF_FLAGS + (size_t)((lane & 16) ? 16 + tj : ti) * 512 + s * 64) + (lane & 15);
    const char* ux = myws + (size_t)(call & 1) * STAGE_BYTES + ((size_t)s * NCH * NBLK + ti) * PIECE;
    const char* uw = myws + (size_t)(call & 1) * STAGE_BYTES + ((size_t)s * NCH * NBLK + 16 + tj) * PIECE;
    char* slot = lds + s * SLOT;
    const unsigned tag = call << 12;
    unsigned fl = 0;
    unsigned todo = 0, done = 0;
    bool dead = false;
    for (int it = 0; it < NCH; ++it) {
      while (!dead && !todo) {
        fl = __hip_atomic_load(flag, __ATOMIC_RELAXED, __HIP_MEMORY_SCOPE_SYSTEM);
        todo = ready_mask(fl, tag) & ~done;
        if (todo) break;
        if (__builtin_amdgcn_s_memrealtime() - t0 > DEADLINE_TICKS) {
          dead = true;
          if (lane == 0) {
            __hip_atomic_store((unsigned*)(myws + OFF_TMO), call, __ATOMIC_RELAXED, __HIP_MEMORY_SCOPE_AGENT);
            __hip_atomic_store(abortw, 1u, __ATOMIC_RELAXED, __HIP_MEMORY_SCOPE_WORKGROUP);
          }
          break;
        }
        __builtin_amdgcn_s_sleep(PSLEEP);
      }
      STAMP(16 + ((wave - 4) & 7) * 8 + it);
      const int c = __builtin_ctz(todo | 0x100u) & (NCH - 1);
      done |= 1u << c;
      if (!dead) {
        const unsigned hx = __builtin_amdgcn_readlane(fl, 2 * c), hw = __builtin_amdgcn_readlane(fl, 16 + 2 * c);
        u32x4 d[4][4]; unsigned li[4], hd[4] = {hx & 63u, (hx >> 6) & 63u, hw & 63u, (hw >> 6) & 63u};
#pragma unroll
        for (int q = 0; q < 4; ++q) {
          const char* p = (q < 2 ? ux : uw) + (size_t)c * (NBLK * PIECE) + (q & 1) * 4096;
          const unsigned o3 = hd[q] >= HDR_RAW ? lane * 16 : lane * 8;
          asm volatile("global_load_dwordx4 %0, %5, %8 sc0 sc1\n\tglobal_load_dwordx4 %1, %5, %8 offset:1024 sc0 sc1\n\t"
                       "global_load_dwordx4 %2, %5, %8 offset:2048 sc0 sc1\n\tglobal_load_dwordx4 %3, %6, %8 offset:3072 sc0 sc1\n\t"
                       "global_load_ushort %4, %7, %8 offset:3584 sc0 sc1"
                       : "=&v"(d[q][0]), "=&v"(d[q][1]), "=&v"(d[q][2]), "=&v"(d[q][3]), "=&v"(li[q])
                       : "v"(lane * 16), "v"(o3), "v"(lane * 2), "s"(p) : "memory");
        }
        asm volatile("global_load_dword %0, %1, off sc0 sc1" : "=&v"(fl) : "v"(flag) : "memory");
        while (__hip_atomic_load(freec + s, __ATOMIC_ACQUIRE, __HIP_MEMORY_SCOPE_WORKGROUP) < 4u * it) __builtin_amdgcn_s_sleep(1);
        asm volatile("s_waitcnt vmcnt(0)" : "+v"(fl), "+v"(d[0][0]), "+v"(d[0][1]), "+v"(d[0][2]), "+v"(d[0][3]), "+v"(d[1][0]), "+v"(d[1][1]), "+v"(d[1][2]), "+v"(d[1][3]),
                     "+v"(d[2][0]), "+v"(d[2][1]), "+v"(d[2][2]), "+v"(d[2][3]), "+v"(d[3][0]), "+v"(d[3][1]), "+v"(d[3][2]), "+v"(d[3][3]),
                     "+v"(li[0]), "+v"(li[1]), "+v"(li[2]), "+v"(li[3]) :: "memory");
        todo = ready_mask(fl, tag) & ~done;
#pragma unroll
        for (int q = 0; q < 4; ++q) {
          if (hd[q] < HDR_RAW) {
            const unsigned bb = hd[q] * 0x04000400u;
            unsigned g0 = 0, g1 = 0;
#pragma unroll
            for (int k = 0; k < 14; ++k) {
              const unsigned w = d[q][k >> 2][k & 3];
              if (k < 7) g0 = bfi(0x60006000u, w, g0 >> 2); else g1 = bfi(0x60006000u, w, g1 >> 2);
              d[q][k >> 2][k & 3] = (w & 0x9FFF9FFFu) + bb;
            }
            d[q][3][2] = (((g0 >> 1) & 0x1FFF1FFFu) | ((g0 << 1) & 0x80008000u)) + bb;
            d[q][3][3] = (((g1 >> 1) & 0x1FFF1FFFu) | ((g1 << 1) & 0x80008000u)) + bb;
          } else li[q] = 0;
#pragma unroll
          for (int f = 0; f < 4; ++f) *(lds_u32x4*)(slot + q * 4096 + f * 1024 + lane * 16) = d[q][f];
        }
        if (c < NS) {
        unsigned short ov[4];
#pragma unroll
        for (int q = 0; q < 4; ++q) ov[q] = *(lds_u16*)(slot + q * 4096 + (li[q] & 0x7FFu) * 2);
#pragma unroll
        for (int q = 0; q < 4; ++q)
          if (li[q] & 0x6000u) *(lds_u16*)(slot + q * 4096 + (li[q] & 0x7FFu) * 2) = (unsigned short)bfi(0x8000u, ov[q], ov[q] + (li[q] & 0x6000u));
        }
      } else
      while (__hip_atomic_load(freec + s, __ATOMIC_ACQUIRE, __HIP_MEMORY_SCOPE_WORKGROUP) < 4u * it) __builtin_amdgcn_s_sleep(1);
      if (lane == 0) __hip_atomic_store(full + s, (unsigned)(it + 1), __ATOMIC_RELEASE, __HIP_MEMORY_SCOPE_WORKGROUP);
    }
    return;
  }

  const int wr = (wave - 1) >> 1, wc = (wave - 1) & 1;
  f32x4 acc[4][4];
#pragma unroll
  for (int m = 0; m < 4; ++m)
#pragma unroll
    for (int u = 0; u < 4; ++u) acc[m][u] = (f32x4){0.f, 0.f, 0.f, 0.f};
  unsigned consumed = 0;
  const char* abase = lds + wr * 4096 + lane * 16;
  const char* bbase = lds + PIECE + wc * 4096 + lane * 16;
  unsigned ready = 0, s = 0;
  for (int it = 0; it < NR * NCH; ++it) {
    while (!ready) {
      unsigned v = __hip_atomic_load(full + (lane & 7), __ATOMIC_ACQUIRE, __HIP_MEMORY_SCOPE_WORKGROUP);
      ready = (unsigned)__builtin_amdgcn_ballot_w64(v > consumed) & 0xFFFFu;
      if (ready) break;
      __builtin_amdgcn_s_sleep(1);
    }
    s = (s + 1 + __builtin_ctz(ready >> (s + 1))) & 7;
    f16x8 a[4], bq[4];
#pragma unroll
    for (int u = 0; u < 4; ++u) bq[u] = *(const f16x8*)(bbase + s * SLOT + u * 1024);
#pragma unroll
    for (int m = 0; m < 4; ++m) a[m] = *(const f16x8*)(abase + s * SLOT + m * 1024);
    unsigned v = *(const volatile __attribute__((address_space(3))) unsigned*)(full + (lane & 7));
    consumed += ((lane & 7) == s);
    PIN();
#pragma unroll
    for (int m = 0; m < 3; ++m)
#pragma unroll
      for (int u = 0; u < 4; ++u) acc[m][u] = __builtin_amdgcn_mfma_f32_16x16x32_f16(a[m], bq[u], acc[m][u], 0, 0, 0);
    PIN();
    asm volatile("s_waitcnt lgkmcnt(0)" : "+v"(a[3]), "+v"(v) :: "memory");
    if (lane == 0) __hip_atomic_fetch_add(freec + s, 1u, __ATOMIC_RELAXED, __HIP_MEMORY_SCOPE_WORKGROUP);
    ready = (unsigned)__builtin_amdgcn_ballot_w64(v > consumed) & 0xFFFFu;
    asm volatile("" ::: "memory");
#pragma unroll
    for (int u = 0; u < 4; ++u) acc[3][u] = __builtin_amdgcn_mfma_f32_16x16x32_f16(a[3], bq[u], acc[3][u], 0, 0, 0);
  }
  STAMP(80 + wave);
  const bool bad = __hip_atomic_load(abortw, __ATOMIC_RELAXED, __HIP_MEMORY_SCOPE_WORKGROUP) != 0;
  float* o = P.out + (size_t)(128 * ti + 64 * wr + 4 * (lane >> 4)) * ND + 128 * tj + 64 * wc + 4 * (lane & 15);
#pragma unroll
  for (int m = 0; m < 4; ++m)
#pragma unroll
    for (int r = 0; r < 4; ++r) {
      f32x4 v;
#pragma unroll
      for (int u = 0; u < 4; ++u) { float y = acc[m][u][r]; v[u] = bad ? __builtin_nanf("") : (y < 0.f ? 0.f : y); }
      asm volatile("global_store_dwordx4 %0, %1, off sc0 sc1 nt\n\ts_nop 1" :: "v"(o + (size_t)(16 * m + r) * ND), "v"(v) : "memory");
    }
}

extern "C" void kernel_launch(int rank, int nranks, ncclComm_t comm, hipStream_t stream,
                              void* const* d_in, const int* in_sizes, int n_in,
                              void* d_out, int out_size, const int* in_dtypes, int out_dtype,
                              void* const* d_out_all, void* const* d_ws_all, size_t ws_bytes) {
  if (nranks != NR || (unsigned)rank >= (unsigned)NR || n_in != 2 || ws_bytes < WS_NEED ||
      in_sizes[0] != MD * KS || in_sizes[1] != KS * ND || out_size != MD * ND) {
    fprintf(stderr, "gemm_ar: unexpected configuration\n");
    return;
  }
  hipError_t e = hipFuncSetAttribute((const void*)gemm_ar_kernel, hipFuncAttributeMaxDynamicSharedMemorySize, LDS_BYTES);
  if (e != hipSuccess) { fprintf(stderr, "hipFuncSetAttribute: %s\n", hipGetErrorString(e)); return; }
  Params P;
  P.x = (const float*)d_in[0]; P.w = (const float*)d_in[1]; P.out = (float*)d_out; P.rank = rank;
  for (int p = 0; p < NR; ++p) P.ws[p] = (char*)d_ws_all[p];
  hipLaunchKernelGGL(gemm_ar_kernel, dim3(256), dim3(NTHREADS), LDS_BYTES, stream, P);
}
```
